```python
import numpy as np

kernel_name = "gpu_gemm_a2a_m4096_k4096_n32768_f32_none"


ABSMAX_RTOL = 0.001

COMPUTE_DTYPE = "float32"
EPILOGUE = "none"
IN_QUANT = None
OUT_QUANT = None
IN_DTYPES = {"x": "float32", "w_mat": "float32"}
WORLD_SIZE = 8

def setup_inputs():

    x = np.random.randn(4096, 4096).astype(np.float32)
    w_mat = (np.random.randn(4096, 32768) / np.sqrt(4096)).astype(np.float32)
    return {"x": x, "w_mat": w_mat}

def _epilogue(y):
    return y

def reference(x, w_mat):
    y = (x @ w_mat).astype(np.float32)
    return _epilogue(y)
```

```cpp
#include <hip/hip_runtime.h>
#include <rccl/rccl.h>
#include <cstdio>

#define HIP_CHECK(expr) do { hipError_t _e = (expr); if (_e != hipSuccess) { \
    fprintf(stderr, "HIP error %s at %s:%d\n", hipGetErrorString(_e), __FILE__, __LINE__); \
    return; } } while (0)

using f16x8 = __attribute__((ext_vector_type(8))) _Float16;
using f16x4 = __attribute__((ext_vector_type(4))) _Float16;
using f32x4 = __attribute__((ext_vector_type(4))) float;

constexpr int NR = 8;
constexpr int MG = 4096, KK = 4096, NL = 4096, NG = 32768, MS = 512;
constexpr int BM = 256, BK = 64, HALF = 128, NXCD = 8, WGM = 8, HT = HALF * BK, SHM_B = 8 * HT * 2;

constexpr size_t WS_CNT0 = 0;
constexpr size_t WS_CNT1 = 128;
constexpr size_t WS_TMO = 256;
constexpr size_t WS_FLAG = 4096;
constexpr size_t WS_A = 1u << 20;
constexpr size_t A_BYTES = (size_t)MG * KK * 2;
constexpr size_t WS_WT = WS_A + 2 * A_BYTES;
constexpr int P0_PUSH = 256;
constexpr int P0_CONV = 1024;
constexpr unsigned long long DEADLINE_TICKS = 100000000ull;

__device__ __forceinline__ int lds_byte(int r, int c) {
  int st = (r >> 4) * 2 + (c >> 5), rr = r & 15, cc = c & 31, ob = rr * 64 + cc * 2;
  return st * 1024 + (ob ^ (((ob >> 9) & 1) << 5));
}
__device__ __forceinline__ void stage_rc(int b, int& R, int& C) {
  int st = b / 1024, sb = b % 1024, swz = sb ^ (((sb >> 9) & 1) << 5);
  R = (st >> 1) * 16 + swz / 64; C = (st & 1) * 32 + (swz % 64) / 2;
}

__device__ __forceinline__ void wait_flag(unsigned* flag, unsigned target, unsigned* tmo, unsigned call) {
  unsigned long long t0 = __builtin_amdgcn_s_memrealtime();
  while (__hip_atomic_load(flag, __ATOMIC_RELAXED, __HIP_MEMORY_SCOPE_SYSTEM) < target) {
    if (__hip_atomic_load(tmo, __ATOMIC_RELAXED, __HIP_MEMORY_SCOPE_AGENT) == call ||
        __builtin_amdgcn_s_memrealtime() - t0 > DEADLINE_TICKS) {
      __hip_atomic_store(tmo, call, __ATOMIC_RELAXED, __HIP_MEMORY_SCOPE_AGENT);
      break;
    }
    __builtin_amdgcn_s_sleep(2);
  }
  __builtin_amdgcn_fence(__ATOMIC_ACQUIRE, "");
}

__global__ __launch_bounds__(512, 2)
void fused_kernel(int phase, int rank, const float* __restrict__ x, const float* __restrict__ W,
                  float* __restrict__ out, char* const* __restrict__ ws_all) {
  extern __shared__ __attribute__((aligned(16))) _Float16 shm[];
  char* ws = ws_all[rank];
  __shared__ unsigned s_call;
  if (threadIdx.x == 0) {
    unsigned* cnt = (unsigned*)(ws + (phase == 0 ? WS_CNT0 : WS_CNT1));
    unsigned t = __hip_atomic_fetch_add(cnt, 1u, __ATOMIC_RELAXED, __HIP_MEMORY_SCOPE_AGENT);
    s_call = t / gridDim.x + 1;
  }
  __syncthreads();
  const unsigned call = s_call;
  const size_t abuf = WS_A + (size_t)(call & 1) * A_BYTES;

  if (phase == 0) {
    if (blockIdx.x < P0_PUSH) {
      const int w = blockIdx.x;
      const float4* src = (const float4*)(x + (size_t)(2 * w) * KK);
      f16x8 v[2];
      for (int i = 0; i < 2; ++i) {
        float4 a = src[(threadIdx.x + i * 512) * 2], b = src[(threadIdx.x + i * 512) * 2 + 1];
        v[i][0] = (_Float16)a.x; v[i][1] = (_Float16)a.y; v[i][2] = (_Float16)a.z; v[i][3] = (_Float16)a.w;
        v[i][4] = (_Float16)b.x; v[i][5] = (_Float16)b.y; v[i][6] = (_Float16)b.z; v[i][7] = (_Float16)b.w;
      }
      for (int pp = 0; pp < NR; ++pp) {
        unsigned p = (unsigned)(rank + pp) % NR;
        f16x8* dst = (f16x8*)(ws_all[p] + abuf + ((size_t)(rank * MS + 2 * w) * KK) * 2);
        dst[threadIdx.x] = v[0];
        dst[threadIdx.x + 512] = v[1];
      }
      asm volatile("s_waitcnt vmcnt(0)" ::: "memory");
      __syncthreads();
      if (threadIdx.x < NR) {
        unsigned p = (unsigned)(rank + threadIdx.x) % NR;
        __builtin_amdgcn_fence(__ATOMIC_RELEASE, "");
        asm volatile("s_waitcnt vmcnt(0)" ::: "memory");
        __hip_atomic_fetch_add((unsigned*)(ws_all[p] + WS_FLAG + 128 * rank), 1u, __ATOMIC_RELAXED,
                               __HIP_MEMORY_SCOPE_SYSTEM);
      }
    } else {
      const int t = blockIdx.x - P0_PUSH;
      const int k0 = (t >> 5) * 128, n0 = (t & 31) * 128;
      constexpr int LP = 136;
      const int tn = (threadIdx.x & 31) * 4, tk = threadIdx.x >> 5;
      for (int i = 0; i < 8; ++i) {
        int k = tk + i * 16;
        float4 a = *(const float4*)(W + (size_t)(k0 + k) * NG + rank * NL + n0 + tn);
        shm[(tn + 0) * LP + k] = (_Float16)a.x;
        shm[(tn + 1) * LP + k] = (_Float16)a.y;
        shm[(tn + 2) * LP + k] = (_Float16)a.z;
        shm[(tn + 3) * LP + k] = (_Float16)a.w;
      }
      __syncthreads();
      _Float16* wt = (_Float16*)(ws + WS_WT);
      for (int i = 0; i < 4; ++i) {
        int e = threadIdx.x + i * 512;
        int n = e >> 4, kc = (e & 15) * 8;
        f16x8 v = *(const f16x8*)(shm + n * LP + kc);
        *(f16x8*)(wt + (size_t)(n0 + n) * KK + k0 + kc) = v;
      }
    }
    return;
  }

  const _Float16* A = (const _Float16*)(ws + abuf);
  const _Float16* Bt = (const _Float16*)(ws + WS_WT);
  float* C = out;
  constexpr int K = KK, N = NL;
  #define SA(b,h) (shm+((b)*2+(h))*HT)
  #define SB(b,h) (shm+(4+(b)*2+(h))*HT)
  #define STAGE_X(T,P,BASE,br,kt) do{long _g=(long)(br)*K+(long)(kt)*BK; \
    for(int _i=0;_i<2;++_i){int _b=(T)*16+_i*8192;int _r,_c;stage_rc(_b,_r,_c); \
      __builtin_amdgcn_global_load_lds((const unsigned*)(BASE+_g+(long)_r*K+_c), \
        (unsigned*)((char*)(P)+_b),16,0,0);}}while(0)
  #define STAGE(P,BASE,br,kt) STAGE_X(threadIdx.x,P,BASE,br,kt)
  #define LDA(dst,b,h) for(int m=0;m<4;++m)for(int k=0;k<2;++k) \
    dst[m][k]=*reinterpret_cast<const f16x8*>((char*)SA(b,h)+lds_byte(wr*64+m*16+fr,k*32+fq*8))
  #define LDB(dst,b,h) for(int n=0;n<2;++n)for(int k=0;k<2;++k) \
    dst[n][k]=*reinterpret_cast<const f16x8*>((char*)SB(b,h)+lds_byte(wc*32+n*16+fr,k*32+fq*8))
  #define MMA(ai,bj,At,Bt) do{__builtin_amdgcn_s_setprio(1); \
    for(int m=0;m<4;++m)for(int n=0;n<2;++n)for(int k=0;k<2;++k) \
      acc[ai][bj][m][n]=__builtin_amdgcn_mfma_f32_16x16x32_f16(At[m][k],Bt[n][k],acc[ai][bj][m][n],0,0,0); \
    __builtin_amdgcn_s_setprio(0);}while(0)
  #define WAIT_V(n) asm volatile("s_waitcnt vmcnt(" #n ")":::"memory")
  #define WAIT_L(n) asm volatile("s_waitcnt lgkmcnt(" #n ")":::"memory")
  #define BAR __builtin_amdgcn_s_barrier()
  #define SCHED __builtin_amdgcn_sched_barrier(0)

  int nM = MG / BM, nN = N / BM, nwg = nM * nN, wgid = blockIdx.x;
  { int q = nwg / NXCD, r = nwg % NXCD, xcd = wgid % NXCD, off = wgid / NXCD;
    wgid = (xcd < r ? xcd * (q + 1) : r * (q + 1) + (xcd - r) * q) + off; }
  int nig = WGM * nN, gid = wgid / nig, fm = gid * WGM, gsz = min(nM - fm, WGM);
  int pm = fm + ((wgid % nig) % gsz), pn = (wgid % nig) / gsz, brow = pm * BM, bcol = pn * BM;
  int wid = threadIdx.x >> 6, lane = threadIdx.x & 63, wr = wid >> 2, wc = wid & 3, fr = lane & 15, fq = lane >> 4;

  unsigned* tmo = (unsigned*)(ws + WS_TMO);
  if (threadIdx.x == 0)
    wait_flag((unsigned*)(ws + WS_FLAG + 128 * (brow / MS)), 256u * call, tmo, call);
  __syncthreads();
  if (__hip_atomic_load(tmo, __ATOMIC_RELAXED, __HIP_MEMORY_SCOPE_AGENT) == call) {
    for (int i = threadIdx.x; i < BM * BM; i += 512)
      C[(long)(brow + i / BM) * N + bcol + (i % BM)] = __builtin_nanf("");
    return;
  }

  f32x4 acc[2][2][4][2] = {};
  f16x8 At[4][2], B0[2][2], B1[2][2];
  int nt = K / BK;
  STAGE(SB(0,0),Bt,bcol,0); STAGE(SA(0,0),A,brow,0);
  STAGE(SB(0,1),Bt,bcol+HALF,0); STAGE(SA(0,1),A,brow+HALF,0);
  if (wr == 1) BAR;
  WAIT_V(4); BAR;
  STAGE(SB(1,0),Bt,bcol,1); STAGE(SA(1,0),A,brow,1); STAGE(SB(1,1),Bt,bcol+HALF,1);
  WAIT_V(6); BAR;
  for (int t = 0; t < nt - 2; t += 2) {
    LDB(B0,0,0); SCHED; LDA(At,0,0); STAGE(SA(1,1),A,brow+HALF,t+1);
    WAIT_L(8); BAR; WAIT_L(0); MMA(0,0,At,B0); BAR; SCHED;
    LDB(B1,0,1); STAGE(SB(0,0),Bt,bcol,t+2);
    BAR; WAIT_L(0); MMA(0,1,At,B1); BAR;
    LDA(At,0,1); STAGE(SA(0,0),A,brow,t+2);
    BAR; WAIT_L(0); MMA(1,0,At,B0); BAR; SCHED;
    STAGE(SB(0,1),Bt,bcol+HALF,t+2);
    WAIT_V(6); BAR; MMA(1,1,At,B1); BAR;
    LDB(B0,1,0); SCHED; LDA(At,1,0); STAGE(SA(0,1),A,brow+HALF,t+2);
    WAIT_L(8); BAR; WAIT_L(0); MMA(0,0,At,B0); BAR; SCHED;
    LDB(B1,1,1); STAGE(SB(1,0),Bt,bcol,t+3);
    BAR; WAIT_L(0); MMA(0,1,At,B1); BAR;
    LDA(At,1,1); STAGE(SA(1,0),A,brow,t+3);
    BAR; WAIT_L(0); MMA(1,0,At,B0); BAR; SCHED;
    STAGE(SB(1,1),Bt,bcol+HALF,t+3);
    WAIT_V(6); BAR; MMA(1,1,At,B1); BAR;
  }
  int tz = threadIdx.x; asm volatile("" : "+v"(tz)); wid = tz >> 6; lane = tz & 63; wr = wid >> 2; wc = wid & 3; fr = lane & 15; fq = lane >> 4;
  { LDB(B0,0,0); WAIT_V(0); LDA(At,0,0); STAGE_X(tz,SA(1,1),A,brow+HALF,nt-1);
    BAR; WAIT_L(0); MMA(0,0,At,B0); BAR;
    LDB(B1,0,1); BAR; WAIT_L(0); MMA(0,1,At,B1); BAR;
    LDA(At,0,1); WAIT_V(4); BAR; WAIT_L(0); MMA(1,0,At,B0); MMA(1,1,At,B1); BAR; }
  { LDB(B0,1,0); LDA(At,1,0); WAIT_V(2); BAR; WAIT_L(0); MMA(0,0,At,B0); BAR;
    LDB(B1,1,1); WAIT_V(0); BAR; WAIT_L(0); MMA(0,1,At,B1); BAR;
    LDA(At,1,1); BAR; WAIT_L(0); MMA(1,0,At,B0); MMA(1,1,At,B1); BAR; }
  if (wr == 0) BAR;
  for (int ai = 0; ai < 2; ++ai) for (int bj = 0; bj < 2; ++bj) for (int m = 0; m < 4; ++m) for (int n = 0; n < 2; ++n) for (int j = 0; j < 4; ++j)
    C[(long)(brow + ai * HALF + wr * 64 + m * 16 + fq * 4 + j) * N + (bcol + bj * HALF + wc * 32 + n * 16 + fr)] = acc[ai][bj][m][n][j];
}

extern "C" void kernel_launch(int rank, int nranks, ncclComm_t comm,
                              hipStream_t stream,
                              void* const* d_in, const int* in_sizes, int n_in,
                              void* d_out, int out_size,
                              const int* in_dtypes, int out_dtype,
                              void* const* d_out_all,
                              void* const* d_ws_all, size_t ws_bytes) {
  static bool init = false;
  if (!init) {
    HIP_CHECK(hipFuncSetAttribute((const void*)fused_kernel, hipFuncAttributeMaxDynamicSharedMemorySize, SHM_B));
    init = true;
  }
  if (nranks != NR || ws_bytes < WS_WT + A_BYTES) return;
  fused_kernel<<<P0_PUSH + P0_CONV, 512, SHM_B, stream>>>(0, rank, (const float*)d_in[0], (const float*)d_in[1],
                                                         (float*)d_out, (char* const*)d_ws_all);
  fused_kernel<<<256, 512, SHM_B, stream>>>(1, rank, (const float*)d_in[0], (const float*)d_in[1],
                                            (float*)d_out, (char* const*)d_ws_all);
}
```

```cpp
#include <hip/hip_runtime.h>
#include <rccl/rccl.h>
#include <cstdio>

#define HIP_CHECK(expr) do { hipError_t _e = (expr); if (_e != hipSuccess) { \
    fprintf(stderr, "HIP error %s at %s:%d\n", hipGetErrorString(_e), __FILE__, __LINE__); \
    return; } } while (0)

using f16x8 = __attribute__((ext_vector_type(8))) _Float16;
using f16x4 = __attribute__((ext_vector_type(4))) _Float16;
using f32x4 = __attribute__((ext_vector_type(4))) float;

constexpr int NR = 8;
constexpr int MG = 4096, KK = 4096, NL = 4096, NG = 32768, MS = 512;
constexpr int BM = 256, BK = 64, HALF = 128, NXCD = 8, WGM = 8, HT = HALF * BK, GEMM_B = 8 * HT * 2, SHM_B = GEMM_B + 16384;

constexpr size_t WS_CNT0 = 0;
constexpr size_t WS_CNT1 = 128;
constexpr size_t WS_TMO = 256;
constexpr size_t WS_FLAG = 4096;
constexpr size_t WS_A = 1u << 20;
constexpr size_t A_BYTES = (size_t)MG * KK * 2;
constexpr size_t WS_WT = WS_A + 2 * A_BYTES;
constexpr int P0_PUSH = 0;
constexpr int P0_CONV = 1024;
constexpr unsigned long long DEADLINE_TICKS = 100000000ull;

__device__ __forceinline__ int lds_byte(int r, int c) {
  int st = (r >> 4) * 2 + (c >> 5), rr = r & 15, cc = c & 31, ob = rr * 64 + cc * 2;
  return st * 1024 + (ob ^ (((ob >> 9) & 1) << 5));
}
__device__ __forceinline__ void stage_rc(int b, int& R, int& C) {
  int st = b / 1024, sb = b % 1024, swz = sb ^ (((sb >> 9) & 1) << 5);
  R = (st >> 1) * 16 + swz / 64; C = (st & 1) * 32 + (swz % 64) / 2;
}

__device__ __forceinline__ void wait_flag(unsigned* flag, unsigned target, unsigned* tmo, unsigned call) {
  unsigned long long t0 = __builtin_amdgcn_s_memrealtime();
  while (__hip_atomic_load(flag, __ATOMIC_RELAXED, __HIP_MEMORY_SCOPE_SYSTEM) < target) {
    if (__hip_atomic_load(tmo, __ATOMIC_RELAXED, __HIP_MEMORY_SCOPE_AGENT) == call ||
        __builtin_amdgcn_s_memrealtime() - t0 > DEADLINE_TICKS) {
      __hip_atomic_store(tmo, call, __ATOMIC_RELAXED, __HIP_MEMORY_SCOPE_AGENT);
      break;
    }
    __builtin_amdgcn_s_sleep(2);
  }
  __builtin_amdgcn_fence(__ATOMIC_ACQUIRE, "");
}

__global__ __launch_bounds__(512, 2)
void fused_kernel(int phase, int rank, const float* __restrict__ x, const float* __restrict__ W,
                  float* __restrict__ out, char* const* __restrict__ ws_all) {
  extern __shared__ __attribute__((aligned(16))) _Float16 shm[];
  char* ws = ws_all[rank];
  __shared__ unsigned s_call;
  if (threadIdx.x == 0) {
    unsigned* cnt = (unsigned*)(ws + (phase == 0 ? WS_CNT0 : WS_CNT1));
    unsigned t = __hip_atomic_fetch_add(cnt, 1u, __ATOMIC_RELAXED, __HIP_MEMORY_SCOPE_AGENT);
    s_call = t / gridDim.x + 1;
  }
  __syncthreads();
  const unsigned call = s_call;
  const size_t abuf = WS_A + (size_t)(call & 1) * A_BYTES;

  if (phase == 0) {
    {
      const int t = blockIdx.x - P0_PUSH;
      const int k0 = (t >> 5) * 128, n0 = (t & 31) * 128;
      constexpr int LP = 136;
      const int tn = (threadIdx.x & 31) * 4, tk = threadIdx.x >> 5;
      for (int i = 0; i < 8; ++i) {
        int k = tk + i * 16;
        float4 a = *(const float4*)(W + (size_t)(k0 + k) * NG + rank * NL + n0 + tn);
        shm[(tn + 0) * LP + k] = (_Float16)a.x;
        shm[(tn + 1) * LP + k] = (_Float16)a.y;
        shm[(tn + 2) * LP + k] = (_Float16)a.z;
        shm[(tn + 3) * LP + k] = (_Float16)a.w;
      }
      __syncthreads();
      _Float16* wt = (_Float16*)(ws + WS_WT);
      for (int i = 0; i < 4; ++i) {
        int e = threadIdx.x + i * 512;
        int n = e >> 4, kc = (e & 15) * 8;
        f16x8 v = *(const f16x8*)(shm + n * LP + kc);
        *(f16x8*)(wt + (size_t)(n0 + n) * KK + k0 + kc) = v;
      }
    }
    return;
  }

  const _Float16* A = (const _Float16*)(ws + abuf);
  const _Float16* Bt = (const _Float16*)(ws + WS_WT);
  float* C = out;
  constexpr int K = KK, N = NL;
  #define SA(b,h) (shm+((b)*2+(h))*HT)
  #define SB(b,h) (shm+(4+(b)*2+(h))*HT)
  #define STAGE_X(T,P,BASE,br,kt) do{long _g=(long)(br)*K+(long)(kt)*BK; \
    for(int _i=0;_i<2;++_i){int _b=(T)*16+_i*8192;int _r,_c;stage_rc(_b,_r,_c); \
      __builtin_amdgcn_global_load_lds((const unsigned*)(BASE+_g+(long)_r*K+_c), \
        (unsigned*)((char*)(P)+_b),16,0,0);}}while(0)
  #define STAGE(P,BASE,br,kt) STAGE_X(threadIdx.x,P,BASE,br,kt)
  #define LDA(dst,b,h) for(int m=0;m<4;++m)for(int k=0;k<2;++k) \
    dst[m][k]=*reinterpret_cast<const f16x8*>((char*)SA(b,h)+lds_byte(wr*64+m*16+fr,k*32+fq*8))
  #define LDB(dst,b,h) for(int n=0;n<2;++n)for(int k=0;k<2;++k) \
    dst[n][k]=*reinterpret_cast<const f16x8*>((char*)SB(b,h)+lds_byte(wc*32+n*16+fr,k*32+fq*8))
  #define MMA(ai,bj,At,Bt) do{__builtin_amdgcn_s_setprio(1); \
    for(int m=0;m<4;++m)for(int n=0;n<2;++n)for(int k=0;k<2;++k) \
      acc[ai][bj][m][n]=__builtin_amdgcn_mfma_f32_16x16x32_f16(At[m][k],Bt[n][k],acc[ai][bj][m][n],0,0,0); \
    __builtin_amdgcn_s_setprio(0);}while(0)
  #define WAIT_V(n) asm volatile("s_waitcnt vmcnt(" #n ")":::"memory")
  #define WAIT_L(n) asm volatile("s_waitcnt lgkmcnt(" #n ")":::"memory")
  #define BAR __builtin_amdgcn_s_barrier()
  #define SCHED __builtin_amdgcn_sched_barrier(0)

  int nM = MG / BM, nN = N / BM, nwg = nM * nN, wgid = blockIdx.x;
  { int q = nwg / NXCD, r = nwg % NXCD, xcd = wgid % NXCD, off = wgid / NXCD;
    wgid = (xcd < r ? xcd * (q + 1) : r * (q + 1) + (xcd - r) * q) + off; }
  int nig = WGM * nN, gid = wgid / nig, fm = gid * WGM, gsz = min(nM - fm, WGM);
  int pm = fm + ((wgid % nig) % gsz), pn = (wgid % nig) / gsz, brow = pm * BM, bcol = pn * BM;
  int wid = threadIdx.x >> 6, lane = threadIdx.x & 63, wr = wid >> 2, wc = wid & 3, fr = lane & 15, fq = lane >> 4;

  unsigned* tmo = (unsigned*)(ws + WS_TMO);
  unsigned* myflags = (unsigned*)(ws + WS_FLAG + (size_t)(brow / MS) * 16 * 128);
  char* xs = (char*)shm + GEMM_B;
  {
    const float4* src = (const float4*)(x + (size_t)(2 * blockIdx.x) * KK);
    for (int i = 0; i < 2; ++i) {
      int e = threadIdx.x + i * 512;
      float4 a = src[e * 2], b = src[e * 2 + 1];
      f16x8 v;
      v[0] = (_Float16)a.x; v[1] = (_Float16)a.y; v[2] = (_Float16)a.z; v[3] = (_Float16)a.w;
      v[4] = (_Float16)b.x; v[5] = (_Float16)b.y; v[6] = (_Float16)b.z; v[7] = (_Float16)b.w;
      *(f16x8*)(xs + e * 16) = v;
    }
  }
  __syncthreads();
  const unsigned peer = (unsigned)(rank + 1 + wid) % NR;
  char* pdst = ws_all[peer] + abuf + ((size_t)(rank * MS + 2 * blockIdx.x + (lane >> 5)) * KK + (lane & 31) * 8) * 2;
  unsigned* pflag = (unsigned*)(ws_all[peer] + WS_FLAG + (size_t)rank * 16 * 128);
  const int plds = (lane >> 5) * 8192 + (lane & 31) * 16;
  #define PUSH(c) do { f16x8 _v = *(const f16x8*)(xs + plds + (c) * 512); \
    asm volatile("global_store_dwordx4 %0, %1, off sc0 sc1\n\ts_nop 1" :: "v"(pdst + (c) * 512), "v"(_v) : "memory"); } while (0)
  #define SIGNAL(c) do { if (lane == 0) __hip_atomic_fetch_add(pflag + (c) * 32, 1u, __ATOMIC_RELAXED, __HIP_MEMORY_SCOPE_SYSTEM); } while (0)
  PUSH(0); PUSH(1); PUSH(2);
  asm volatile("s_waitcnt vmcnt(0)" ::: "memory");
  SIGNAL(0); SIGNAL(1); SIGNAL(2);
  if (threadIdx.x == 0) wait_flag(myflags, 256u * call, tmo, call);
  __syncthreads();

  f32x4 acc[2][2][4][2] = {};
  f16x8 At[4][2], B0[2][2], B1[2][2];
  int nt = K / BK;
  STAGE(SB(0,0),Bt,bcol,0); STAGE(SA(0,0),A,brow,0);
  STAGE(SB(0,1),Bt,bcol+HALF,0); STAGE(SA(0,1),A,brow+HALF,0);
  if (wr == 1) BAR;
  WAIT_V(4); BAR;
  STAGE(SB(1,0),Bt,bcol,1); STAGE(SA(1,0),A,brow,1); STAGE(SB(1,1),Bt,bcol+HALF,1);
  WAIT_V(6); BAR;
  for (int t = 0; t < nt - 2; t += 2) {
    const int cn = (t + 2) >> 2;
    const bool bnd = (t & 3) == 2;
    const bool extra = bnd ? (cn + 2 < 16) : (t >= 4 && (t >> 2) + 2 < 16);
    if (bnd && threadIdx.x == 0) wait_flag(myflags + cn * 32, 256u * call, tmo, call);
    LDB(B0,0,0); SCHED; LDA(At,0,0); STAGE(SA(1,1),A,brow+HALF,t+1);
    WAIT_L(8); BAR; WAIT_L(0); MMA(0,0,At,B0); BAR; SCHED;
    LDB(B1,0,1); STAGE(SB(0,0),Bt,bcol,t+2);
    BAR; WAIT_L(0); MMA(0,1,At,B1); BAR;
    LDA(At,0,1); STAGE(SA(0,0),A,brow,t+2);
    BAR; WAIT_L(0); MMA(1,0,At,B0); BAR; SCHED;
    STAGE(SB(0,1),Bt,bcol+HALF,t+2);
    WAIT_V(6); BAR; MMA(1,1,At,B1); BAR;
    LDB(B0,1,0); SCHED; LDA(At,1,0); STAGE(SA(0,1),A,brow+HALF,t+2);
    if (extra) { if (bnd) PUSH(cn + 2); else SIGNAL((t >> 2) + 2); }
    WAIT_L(8); BAR; WAIT_L(0); MMA(0,0,At,B0); BAR; SCHED;
    LDB(B1,1,1); STAGE(SB(1,0),Bt,bcol,t+3);
    BAR; WAIT_L(0); MMA(0,1,At,B1); BAR;
    LDA(At,1,1); STAGE(SA(1,0),A,brow,t+3);
    BAR; WAIT_L(0); MMA(1,0,At,B0); BAR; SCHED;
    STAGE(SB(1,1),Bt,bcol+HALF,t+3);
    if (extra) WAIT_V(7); else WAIT_V(6);
    BAR; MMA(1,1,At,B1); BAR;
  }
  int tz = threadIdx.x; asm volatile("" : "+v"(tz)); wid = tz >> 6; lane = tz & 63; wr = wid >> 2; wc = wid & 3; fr = lane & 15; fq = lane >> 4;
  { LDB(B0,0,0); WAIT_V(0); LDA(At,0,0); STAGE_X(tz,SA(1,1),A,brow+HALF,nt-1);
    BAR; WAIT_L(0); MMA(0,0,At,B0); BAR;
    LDB(B1,0,1); BAR; WAIT_L(0); MMA(0,1,At,B1); BAR;
    LDA(At,0,1); WAIT_V(4); BAR; WAIT_L(0); MMA(1,0,At,B0); MMA(1,1,At,B1); BAR; }
  { LDB(B0,1,0); LDA(At,1,0); WAIT_V(2); BAR; WAIT_L(0); MMA(0,0,At,B0); BAR;
    LDB(B1,1,1); WAIT_V(0); BAR; WAIT_L(0); MMA(0,1,At,B1); BAR;
    LDA(At,1,1); BAR; WAIT_L(0); MMA(1,0,At,B0); MMA(1,1,At,B1); BAR; }
  if (wr == 0) BAR;
  if (__hip_atomic_load(tmo, __ATOMIC_RELAXED, __HIP_MEMORY_SCOPE_AGENT) == call) {
    for (int i = tz; i < BM * BM; i += 512)
      C[(long)(brow + i / BM) * N + bcol + (i % BM)] = __builtin_nanf("");
    return;
  }
  for (int ai = 0; ai < 2; ++ai) for (int bj = 0; bj < 2; ++bj) for (int m = 0; m < 4; ++m) for (int n = 0; n < 2; ++n) for (int j = 0; j < 4; ++j)
    C[(long)(brow + ai * HALF + wr * 64 + m * 16 + fq * 4 + j) * N + (bcol + bj * HALF + wc * 32 + n * 16 + fr)] = acc[ai][bj][m][n][j];
}

extern "C" void kernel_launch(int rank, int nranks, ncclComm_t comm,
                              hipStream_t stream,
                              void* const* d_in, const int* in_sizes, int n_in,
                              void* d_out, int out_size,
                              const int* in_dtypes, int out_dtype,
                              void* const* d_out_all,
                              void* const* d_ws_all, size_t ws_bytes) {
  static bool init = false;
  if (!init) {
    HIP_CHECK(hipFuncSetAttribute((const void*)fused_kernel, hipFuncAttributeMaxDynamicSharedMemorySize, SHM_B));
    init = true;
  }
  if (nranks != NR || ws_bytes < WS_WT + A_BYTES) return;
  fused_kernel<<<P0_PUSH + P0_CONV, 512, SHM_B, stream>>>(0, rank, (const float*)d_in[0], (const float*)d_in[1],
                                                         (float*)d_out, (char* const*)d_ws_all);
  fused_kernel<<<256, 512, SHM_B, stream>>>(1, rank, (const float*)d_in[0], (const float*)d_in[1],
                                            (float*)d_out, (char* const*)d_ws_all);
}
```
